```python
import numpy as np

kernel_name = "gpu_gemm_rs_m2048_k2048_n2048_bf16_none"


ABSMAX_RTOL = 0.008

COMPUTE_DTYPE = "bfloat16"
EPILOGUE = "none"
IN_QUANT = None
OUT_QUANT = None
IN_DTYPES = {"x": "bfloat16", "w_mat": "bfloat16"}
WORLD_SIZE = 8

def _to_bf16(x):
    u = np.ascontiguousarray(x, dtype=np.float32).view(np.uint32)
    return ((u + 0x7FFF + ((u >> 16) & 1)) & 0xFFFF0000).view(np.float32)

def setup_inputs():

    x = _to_bf16(np.random.randn(2048, 2048).astype(np.float32))
    w_mat = _to_bf16((np.random.randn(2048, 2048) / np.sqrt(2048)).astype(np.float32))
    return {"x": x, "w_mat": w_mat}

def _epilogue(y):
    return y

def reference(x, w_mat):
    y = (x @ w_mat).astype(np.float32)
    return _epilogue(y)
```

```cpp
#include <hip/hip_runtime.h>
#include <rccl/rccl.h>
#include <cstdio>
#include <cstdint>

#define HIP_CHECK(expr) do { hipError_t _e = (expr); if (_e != hipSuccess) { \
    fprintf(stderr, "HIP error %s at %s:%d\n", hipGetErrorString(_e), __FILE__, __LINE__); \
    return; } } while (0)

typedef __attribute__((ext_vector_type(8))) short bf16x8;
typedef __attribute__((ext_vector_type(4))) short bf16x4;
typedef __attribute__((ext_vector_type(4))) float f32x4;
typedef __attribute__((ext_vector_type(4))) unsigned u32x4;
typedef __attribute__((ext_vector_type(2))) unsigned u32x2;

namespace {

constexpr int NR = 8;
constexpr int MFULL = 2048;
constexpr int NFULL = 2048;
constexpr int KSH = 256;
constexpr int MSH = MFULL / NR;
constexpr int TM = 32, TN = 64;
constexpr int NWG = (MSH / TM) * (NFULL / TN);
constexpr int TILE_BYTES = TM * TN * 2;
constexpr size_t SLOT_BYTES = (size_t)MSH * NFULL * 2;
constexpr size_t STAGE_BYTES = 2 * NR * SLOT_BYTES;
constexpr size_t FLAG_OFF = STAGE_BYTES;
constexpr size_t FLAG_BYTES = (size_t)NWG * 128;
constexpr size_t CNT_OFF = FLAG_OFF + FLAG_BYTES;
constexpr size_t CNT_BYTES = (size_t)NWG * 128;
constexpr size_t TMO_OFF = CNT_OFF + CNT_BYTES;
constexpr size_t WS_NEEDED = TMO_OFF + 128;
constexpr unsigned long long DEADLINE_TICKS = 100000000ull;

struct Params {
    const uint16_t* x;
    const uint16_t* w;
    uint16_t* out;
    char* ws[NR];
    int rank;
};

__device__ __forceinline__ unsigned pack_bf16(float lo, float hi) {
    __bf16 a = (__bf16)lo, b = (__bf16)hi;
    return (unsigned)__builtin_bit_cast(unsigned short, a) |
           ((unsigned)__builtin_bit_cast(unsigned short, b) << 16);
}

__global__ __launch_bounds__(256, 1) void gemm_rs_kernel(Params p) {
    __shared__ __attribute__((aligned(16))) char sW[KSH * TN * 2];
    __shared__ unsigned sState[4];

    const int b = blockIdx.x;
    const int tid = threadIdx.x;
    const int lane = tid & 63;
    const int wave = __builtin_amdgcn_readfirstlane(tid >> 6);
    const int wr = wave >> 1, wc = wave & 1;
    const int rank = p.rank;
    const int tn = (b & 7) * 4 + ((b >> 3) & 3);
    const int tm = b >> 5;
    const int tile = tm * (NFULL / TN) + tn;

    char* myws = p.ws[rank];
    unsigned* cnt = (unsigned*)(myws + CNT_OFF + (size_t)b * 128);
    unsigned* tmo = (unsigned*)(myws + TMO_OFF);
    const unsigned call = __hip_atomic_load(cnt, __ATOMIC_RELAXED, __HIP_MEMORY_SCOPE_AGENT) + 1u;

    {
        u32x4 v[8];
#pragma unroll
        for (int i = 0; i < 8; ++i) {
            const int c = tid + 256 * i, row = c >> 3, ch = c & 7;
            v[i] = *(const u32x4*)(p.w + (size_t)row * NFULL + tn * TN + ch * 8);
        }
#pragma unroll
        for (int i = 0; i < 8; ++i) {
            const int c = tid + 256 * i, row = c >> 3, ch = c & 7;
            *(u32x4*)(sW + row * 128 + ((ch ^ (((row >> 1) & 1) << 2)) << 4)) = v[i];
        }
    }
    const int fr = lane & 15, fg = lane >> 4;
    const int rot = b % 7;
    const uint16_t* xbase = p.x + (size_t)(tm * TM + wr * 16 + fr) * KSH + fg * 8;
    bf16x8 xa[8], xb[8];
    {
        const int d0 = (rank + 1 + rot) & 7;
        const uint16_t* xp = xbase + (size_t)d0 * MSH * KSH;
#pragma unroll
        for (int ks = 0; ks < 8; ++ks) xa[ks] = *(const bf16x8*)(xp + ks * 32);
    }
    __syncthreads();
    if (tid == 0) __hip_atomic_store(cnt, call, __ATOMIC_RELAXED, __HIP_MEMORY_SCOPE_AGENT);

    bf16x8 wf[2][8];
    {
        const unsigned sbase = (unsigned)(size_t)sW;
        const int q = (lane & 15) >> 2, pp = lane & 3;
#pragma unroll
        for (int ks = 0; ks < 8; ++ks) {
#pragma unroll
            for (int t = 0; t < 2; ++t) {
                bf16x4 lo, hi;
                const int k0 = 32 * ks + 8 * fg + q;
                const int ch = 4 * wc + pp;
                const unsigned a0 = sbase + k0 * 128 + ((ch ^ (((k0 >> 1) & 1) << 2)) << 4) + 8 * t;
                const unsigned a1 = a0 + 4 * 128;
                const int k1 = k0 + 4;
                const unsigned a1x = sbase + k1 * 128 + ((ch ^ (((k1 >> 1) & 1) << 2)) << 4) + 8 * t;
                (void)a1;
                asm volatile("ds_read_b64_tr_b16 %0, %2\n\tds_read_b64_tr_b16 %1, %3\n\ts_waitcnt lgkmcnt(0)"
                             : "=&v"(lo), "=&v"(hi) : "v"(a0), "v"(a1x) : "memory");
                wf[t][ks] = __builtin_shufflevector(lo, hi, 0, 1, 2, 3, 4, 5, 6, 7);
            }
        }
    }

    const size_t slot_off = ((size_t)(call & 1u) * NR + rank) * SLOT_BYTES + (size_t)tile * TILE_BYTES +
                            (size_t)wave * 1024 + (size_t)lane * 16;

    f32x4 acc0, acc1;
#pragma unroll
    for (int i = 0; i < 8; ++i) {
        if (i < 7) {
            const int dn = (i + 1 < 7) ? ((rank + 1 + (rot + i + 1) % 7) & 7) : rank;
            const uint16_t* xp = xbase + (size_t)dn * MSH * KSH;
            if (i & 1) {
#pragma unroll
                for (int ks = 0; ks < 8; ++ks) xa[ks] = *(const bf16x8*)(xp + ks * 32);
            } else {
#pragma unroll
                for (int ks = 0; ks < 8; ++ks) xb[ks] = *(const bf16x8*)(xp + ks * 32);
            }
        }
        acc0 = (f32x4){0.f, 0.f, 0.f, 0.f};
        acc1 = (f32x4){0.f, 0.f, 0.f, 0.f};
#pragma unroll
        for (int ks = 0; ks < 8; ++ks) {
            const bf16x8 xf = (i & 1) ? xb[ks] : xa[ks];
            acc0 = __builtin_amdgcn_mfma_f32_16x16x32_bf16(wf[0][ks], xf, acc0, 0, 0, 0);
            acc1 = __builtin_amdgcn_mfma_f32_16x16x32_bf16(wf[1][ks], xf, acc1, 0, 0, 0);
        }
        if (i < 7) {
            const int d = (rank + 1 + (rot + i) % 7) & 7;
            u32x4 pk;
            pk[0] = pack_bf16(acc0[0], acc0[1]);
            pk[1] = pack_bf16(acc0[2], acc0[3]);
            pk[2] = pack_bf16(acc1[0], acc1[1]);
            pk[3] = pack_bf16(acc1[2], acc1[3]);
            *(u32x4*)(p.ws[d] + slot_off) = pk;
        }
    }

    asm volatile("s_waitcnt vmcnt(0)" ::: "memory");
    __syncthreads();
    if (wave == 0) {
        if (lane < NR && lane != rank) {
            __builtin_amdgcn_fence(__ATOMIC_RELEASE, "");
            asm volatile("s_waitcnt vmcnt(0)" ::: "memory");
            unsigned* flag = (unsigned*)(p.ws[lane] + FLAG_OFF + (size_t)b * 128) + rank;
            __hip_atomic_store(flag, call, __ATOMIC_RELAXED, __HIP_MEMORY_SCOPE_SYSTEM);
        }
        bool bad = false;
        if (lane < NR && lane != rank) {
            const unsigned* flag = (const unsigned*)(myws + FLAG_OFF + (size_t)b * 128) + lane;
            const unsigned long long t0 = __builtin_amdgcn_s_memrealtime();
            while (__hip_atomic_load(flag, __ATOMIC_RELAXED, __HIP_MEMORY_SCOPE_SYSTEM) < call) {
                if (__hip_atomic_load(tmo, __ATOMIC_RELAXED, __HIP_MEMORY_SCOPE_AGENT) == call ||
                    __builtin_amdgcn_s_memrealtime() - t0 > DEADLINE_TICKS) {
                    __hip_atomic_store(tmo, call, __ATOMIC_RELAXED, __HIP_MEMORY_SCOPE_AGENT);
                    bad = true;
                    break;
                }
                __builtin_amdgcn_s_sleep(1);
            }
        }
        __builtin_amdgcn_fence(__ATOMIC_ACQUIRE, "");
        asm volatile("s_waitcnt vmcnt(0)" ::: "memory");
        const bool any_bad = __any(bad) ||
            __hip_atomic_load(tmo, __ATOMIC_RELAXED, __HIP_MEMORY_SCOPE_AGENT) == call;
        if (lane == 0) sState[0] = any_bad ? 1u : 0u;
    }
    __syncthreads();
    const bool failed = sState[0] != 0u;

    float r[8] = {acc0[0], acc0[1], acc0[2], acc0[3], acc1[0], acc1[1], acc1[2], acc1[3]};
    if (!failed) {
        const char* sbase = myws + (size_t)(call & 1u) * NR * SLOT_BYTES + (size_t)tile * TILE_BYTES +
                            (size_t)wave * 1024 + (size_t)lane * 16;
        u32x4 pv[NR];
#pragma unroll
        for (int s = 0; s < NR; ++s)
            if (s != rank) pv[s] = *(const u32x4*)(sbase + (size_t)s * SLOT_BYTES);
#pragma unroll
        for (int s = 0; s < NR; ++s)
            if (s != rank) {
#pragma unroll
                for (int j = 0; j < 4; ++j) {
                    r[2 * j] += __builtin_bit_cast(float, pv[s][j] << 16);
                    r[2 * j + 1] += __builtin_bit_cast(float, pv[s][j] & 0xffff0000u);
                }
            }
    } else {
#pragma unroll
        for (int j = 0; j < 8; ++j) r[j] = __builtin_nanf("");
    }
    u32x4 o;
    o[0] = pack_bf16(r[0], r[1]);
    o[1] = pack_bf16(r[2], r[3]);
    o[2] = pack_bf16(r[4], r[5]);
    o[3] = pack_bf16(r[6], r[7]);
    *(u32x4*)(p.out + (size_t)(tm * TM + wr * 16 + fr) * NFULL + tn * TN + wc * 32 + fg * 8) = o;
}

}

extern "C" void kernel_launch(int rank, int nranks, ncclComm_t comm,
                              hipStream_t stream,
                              void* const* d_in, const int* in_sizes, int n_in,
                              void* d_out, int out_size,
                              const int* in_dtypes, int out_dtype,
                              void* const* d_out_all,
                              void* const* d_ws_all, size_t ws_bytes) {
    (void)comm; (void)in_dtypes; (void)out_dtype; (void)d_out_all;
    if (nranks != NR || (unsigned)rank >= (unsigned)NR || n_in < 2 || ws_bytes < WS_NEEDED ||
        in_sizes[0] != MFULL * KSH || in_sizes[1] != KSH * NFULL || out_size != MSH * NFULL) {
        fprintf(stderr, "gemm_rs: unexpected problem shape\n");
        return;
    }
    Params p;
    p.x = (const uint16_t*)d_in[0];
    p.w = (const uint16_t*)d_in[1];
    p.out = (uint16_t*)d_out;
    for (int r = 0; r < NR; ++r) p.ws[r] = (char*)d_ws_all[r];
    p.rank = rank;
    gemm_rs_kernel<<<dim3(NWG), dim3(256), 0, stream>>>(p);
    HIP_CHECK(hipGetLastError());
}
```

```cpp
#include <hip/hip_runtime.h>
#include <rccl/rccl.h>
#include <cstdio>
#include <cstdint>

#define HIP_CHECK(expr) do { hipError_t _e = (expr); if (_e != hipSuccess) { \
    fprintf(stderr, "HIP error %s at %s:%d\n", hipGetErrorString(_e), __FILE__, __LINE__); \
    return; } } while (0)

typedef __attribute__((ext_vector_type(8))) short bf16x8;
typedef __attribute__((ext_vector_type(4))) short bf16x4;
typedef __attribute__((ext_vector_type(4))) float f32x4;
typedef __attribute__((ext_vector_type(4))) unsigned u32x4;

#define PF 2

namespace {

constexpr int NR = 8;
constexpr int MFULL = 2048;
constexpr int NFULL = 2048;
constexpr int KSH = 256;
constexpr int MSH = MFULL / NR;
constexpr int TM = 32, TN = 64;
constexpr int NWG = (MSH / TM) * (NFULL / TN);
constexpr int TILE_BYTES = TM * TN * 2;
constexpr size_t SLOT_BYTES = (size_t)MSH * NFULL * 2;
constexpr size_t STAGE_BYTES = 2 * NR * SLOT_BYTES;
constexpr size_t CNT_OFF = STAGE_BYTES;
constexpr size_t CNT_BYTES = (size_t)NWG * 128;
constexpr size_t TMO_OFF = CNT_OFF + CNT_BYTES;
constexpr size_t WS_NEEDED = TMO_OFF + 128;
constexpr int AUX_SYS = 17;
#define TS(i) do {} while (0)
#define TSH(i) do {} while (0)
constexpr unsigned long long DEADLINE_TICKS = 100000000ull;

struct Params {
    const uint16_t* x;
    const uint16_t* w;
    uint16_t* out;
    char* ws[NR];
    int rank;
};

__device__ __forceinline__ unsigned pack_bf16(float lo, float hi) {
    __bf16 a = (__bf16)lo, b = (__bf16)hi;
    return (unsigned)__builtin_bit_cast(unsigned short, a) |
           ((unsigned)__builtin_bit_cast(unsigned short, b) << 16);
}
__device__ __forceinline__ unsigned encode2(unsigned v) {
    unsigned e = v ^ 0x80008000u;
    e |= ((e & 0x0000ffffu) == 0u) ? 0x00008000u : 0u;
    e |= ((e & 0xffff0000u) == 0u) ? 0x80000000u : 0u;
    return e;
}
__device__ __forceinline__ bool present(u32x4 v) {
    bool ok = true;
#pragma unroll
    for (int j = 0; j < 4; ++j) ok = ok && (v[j] & 0xffffu) != 0u && (v[j] >> 16) != 0u;
    return ok;
}

__global__ __launch_bounds__(256, 1) void gemm_rs_kernel(Params p) {
    __shared__ __attribute__((aligned(16))) char sW[KSH * TN * 2];

    TS(0);
    const int b = blockIdx.x;
    const int tid = threadIdx.x;
    const int lane = tid & 63;
    const int wave = __builtin_amdgcn_readfirstlane(tid >> 6);
    const int wr = wave >> 1, wc = wave & 1;
    const int rank = p.rank;
    const int tn = (b & 7) * 4 + ((b >> 3) & 3);
    const int tm = b >> 5;
    const int tile = tm * (NFULL / TN) + tn;

    char* myws = p.ws[rank];
    unsigned* cnt = (unsigned*)(myws + CNT_OFF + (size_t)b * 128);
    unsigned* tmo = (unsigned*)(myws + TMO_OFF);
    const unsigned call = __hip_atomic_load(cnt, __ATOMIC_RELAXED, __HIP_MEMORY_SCOPE_AGENT) + 1u;
    const __amdgpu_buffer_rsrc_t myrs = __builtin_amdgcn_make_buffer_rsrc(myws, 0, (int)STAGE_BYTES, 0x00020000);
    const unsigned pkt = (unsigned)(tile * TILE_BYTES + wave * 1024 + lane * 16);

    const int fr = lane & 15, fg = lane >> 4;
    const int rot = b % 7;
    const uint16_t* xbase = p.x + (size_t)(tm * TM + wr * 16 + fr) * KSH + fg * 8;
    bf16x8 xf[8][8];
#define LOAD_X(i) do { \
        const int d_ = ((i) < 7) ? ((rank + 1 + (rot + (i)) % 7) & 7) : rank; \
        const uint16_t* xp_ = xbase + (size_t)d_ * MSH * KSH; \
        _Pragma("unroll") for (int ks = 0; ks < 8; ++ks) xf[i][ks] = *(const bf16x8*)(xp_ + ks * 32); \
    } while (0)
    {
        u32x4 v[8];
#pragma unroll
        for (int i = 0; i < 8; ++i) {
            const int c = tid + 256 * i, row = c >> 3, ch = c & 7;
            v[i] = *(const u32x4*)(p.w + (size_t)row * NFULL + tn * TN + ch * 8);
        }
#pragma unroll
        for (int i = 0; i < PF; ++i) LOAD_X(i);
        __builtin_amdgcn_sched_barrier(0);
#pragma unroll
        for (int i = 0; i < 8; ++i) {
            const int c = tid + 256 * i, row = c >> 3, ch = c & 7;
            *(u32x4*)(sW + row * 128 + ((ch ^ (((row >> 1) & 1) << 2)) << 4)) = v[i];
        }
    }
    __syncthreads();
    TS(1);
    TSH(0);
    if (tid == 0) __hip_atomic_store(cnt, call, __ATOMIC_RELAXED, __HIP_MEMORY_SCOPE_AGENT);

    bf16x8 wf[2][8];
    {
        const unsigned sbase = (unsigned)(size_t)sW;
        const int q = (lane & 15) >> 2, pp = lane & 3;
#pragma unroll
        for (int ks = 0; ks < 8; ++ks) {
#pragma unroll
            for (int t = 0; t < 2; ++t) {
                bf16x4 lo, hi;
                const int k0 = 32 * ks + 8 * fg + q;
                const int k1 = k0 + 4;
                const int ch = 4 * wc + pp;
                const unsigned a0 = sbase + k0 * 128 + ((ch ^ (((k0 >> 1) & 1) << 2)) << 4) + 8 * t;
                const unsigned a1 = sbase + k1 * 128 + ((ch ^ (((k1 >> 1) & 1) << 2)) << 4) + 8 * t;
                asm volatile("ds_read_b64_tr_b16 %0, %2\n\tds_read_b64_tr_b16 %1, %3\n\ts_waitcnt lgkmcnt(0)"
                             : "=&v"(lo), "=&v"(hi) : "v"(a0), "v"(a1) : "memory");
                wf[t][ks] = __builtin_shufflevector(lo, hi, 0, 1, 2, 3, 4, 5, 6, 7);
            }
        }
    }

    const unsigned slot_off = (unsigned)(((call & 1u) * NR + rank) * SLOT_BYTES) + pkt;

    f32x4 acc0, acc1;
#pragma unroll
    for (int i = 0; i < 8; ++i) {
        __builtin_amdgcn_sched_barrier(0);
        if (i + PF < 8) LOAD_X(i + PF);
        __builtin_amdgcn_sched_barrier(0);
        acc0 = (f32x4){0.f, 0.f, 0.f, 0.f};
        acc1 = (f32x4){0.f, 0.f, 0.f, 0.f};
#pragma unroll
        for (int ks = 0; ks < 8; ++ks) {
            acc0 = __builtin_amdgcn_mfma_f32_16x16x32_bf16(wf[0][ks], xf[i][ks], acc0, 0, 0, 0);
            acc1 = __builtin_amdgcn_mfma_f32_16x16x32_bf16(wf[1][ks], xf[i][ks], acc1, 0, 0, 0);
        }
        if (i < 7) {
            const int d = (rank + 1 + (rot + i) % 7) & 7;
            u32x4 pk;
            pk[0] = encode2(pack_bf16(acc0[0], acc0[1]));
            pk[1] = encode2(pack_bf16(acc0[2], acc0[3]));
            pk[2] = encode2(pack_bf16(acc1[0], acc1[1]));
            pk[3] = encode2(pack_bf16(acc1[2], acc1[3]));
            const __amdgpu_buffer_rsrc_t rs = __builtin_amdgcn_make_buffer_rsrc(p.ws[d], 0, (int)STAGE_BYTES, 0x00020000);
            __builtin_amdgcn_raw_buffer_store_b128(pk, rs, slot_off, 0, AUX_SYS);
        }
    }
    TS(2);

    u32x4 pv[7];
    bool failed = false;
    {
        const unsigned roff = (unsigned)((call & 1u) * NR * SLOT_BYTES) + pkt;
        unsigned pend = 0x7fu;
        const unsigned long long t0 = __builtin_amdgcn_s_memrealtime();
        for (unsigned spin = 0;; ++spin) {
#pragma unroll
            for (int i = 0; i < 7; ++i)
                if (pend & (1u << i))
                    pv[i] = __builtin_amdgcn_raw_buffer_load_b128(myrs, roff + (unsigned)(((rank + 1 + i) & 7) * SLOT_BYTES), 0, AUX_SYS);
#pragma unroll
            for (int i = 0; i < 7; ++i)
                if ((pend & (1u << i)) && __all(present(pv[i]))) pend &= ~(1u << i);
            if (pend == 0u) break;
            if ((spin & 63u) == 63u) {
                if (__hip_atomic_load(tmo, __ATOMIC_RELAXED, __HIP_MEMORY_SCOPE_AGENT) == call ||
                    __builtin_amdgcn_s_memrealtime() - t0 > DEADLINE_TICKS) {
                    __hip_atomic_store(tmo, call, __ATOMIC_RELAXED, __HIP_MEMORY_SCOPE_AGENT);
                    failed = true;
                    break;
                }
            }
            __builtin_amdgcn_s_sleep(1);
        }
    }
    TS(5);

    float r[8] = {acc0[0], acc0[1], acc0[2], acc0[3], acc1[0], acc1[1], acc1[2], acc1[3]};
    if (!failed) {
#pragma unroll
        for (int i = 0; i < 7; ++i) {
#pragma unroll
            for (int j = 0; j < 4; ++j) {
                const unsigned v = pv[i][j] ^ 0x80008000u;
                r[2 * j] += __builtin_bit_cast(float, v << 16);
                r[2 * j + 1] += __builtin_bit_cast(float, v & 0xffff0000u);
            }
        }
    } else {
#pragma unroll
        for (int j = 0; j < 8; ++j) r[j] = __builtin_nanf("");
    }
    u32x4 o;
    o[0] = pack_bf16(r[0], r[1]);
    o[1] = pack_bf16(r[2], r[3]);
    o[2] = pack_bf16(r[4], r[5]);
    o[3] = pack_bf16(r[6], r[7]);
    *(u32x4*)(p.out + (size_t)(tm * TM + wr * 16 + fr) * NFULL + tn * TN + wc * 32 + fg * 8) = o;
    {
        const unsigned roff = (unsigned)((call & 1u) * NR * SLOT_BYTES) + pkt;
        const u32x4 z = {0u, 0u, 0u, 0u};
#pragma unroll
        for (int i = 0; i < 7; ++i)
            __builtin_amdgcn_raw_buffer_store_b128(z, myrs, roff + (unsigned)(((rank + 1 + i) & 7) * SLOT_BYTES), 0, AUX_SYS);
    }
    TS(7);
    TSH(1);
}

}

extern "C" void kernel_launch(int rank, int nranks, ncclComm_t comm,
                              hipStream_t stream,
                              void* const* d_in, const int* in_sizes, int n_in,
                              void* d_out, int out_size,
                              const int* in_dtypes, int out_dtype,
                              void* const* d_out_all,
                              void* const* d_ws_all, size_t ws_bytes) {
    (void)comm; (void)in_dtypes; (void)out_dtype; (void)d_out_all;
    if (nranks != NR || (unsigned)rank >= (unsigned)NR || n_in < 2 || ws_bytes < WS_NEEDED ||
        in_sizes[0] != MFULL * KSH || in_sizes[1] != KSH * NFULL || out_size != MSH * NFULL) {
        fprintf(stderr, "gemm_rs: unexpected problem shape\n");
        return;
    }
    Params p;
    p.x = (const uint16_t*)d_in[0];
    p.w = (const uint16_t*)d_in[1];
    p.out = (uint16_t*)d_out;
    for (int r = 0; r < NR; ++r) p.ws[r] = (char*)d_ws_all[r];
    p.rank = rank;
    gemm_rs_kernel<<<dim3(NWG), dim3(256), 0, stream>>>(p);
    HIP_CHECK(hipGetLastError());
}
```

```cpp
#include <hip/hip_runtime.h>
#include <rccl/rccl.h>
#include <cstdio>
#include <cstdint>

#define HIP_CHECK(expr) do { hipError_t _e = (expr); if (_e != hipSuccess) { \
    fprintf(stderr, "HIP error %s at %s:%d\n", hipGetErrorString(_e), __FILE__, __LINE__); \
    return; } } while (0)

typedef __attribute__((ext_vector_type(8))) short bf16x8;
typedef __attribute__((ext_vector_type(4))) short bf16x4;
typedef __attribute__((ext_vector_type(4))) float f32x4;
typedef __attribute__((ext_vector_type(4))) unsigned u32x4;

namespace {

constexpr int NR = 8;
constexpr int MFULL = 2048;
constexpr int NFULL = 2048;
constexpr int KSH = 256;
constexpr int MSH = MFULL / NR;
constexpr int TM = 32, TN = 64;
constexpr int NWG = (MSH / TM) * (NFULL / TN);
constexpr int TILE_BYTES = TM * TN * 2;
constexpr size_t SLOT_BYTES = (size_t)MSH * NFULL * 2;
constexpr size_t STAGE_BYTES = 2 * NR * SLOT_BYTES;
constexpr size_t CNT_OFF = STAGE_BYTES;
constexpr size_t CNT_BYTES = (size_t)NWG * 128;
constexpr size_t TMO_OFF = CNT_OFF + CNT_BYTES;
constexpr size_t WS_NEEDED = TMO_OFF + 128;
constexpr int AUX_SYS = 17;
#define TS(i) do {} while (0)
#define TSH(i) do {} while (0)
#define TSK(t) do {} while (0)
#define TSP(j) do {} while (0)
constexpr unsigned long long DEADLINE_TICKS = 100000000ull;

struct Params {
    const uint16_t* x;
    const uint16_t* w;
    uint16_t* out;
    char* ws[NR];
    int rank;
};

__device__ __forceinline__ unsigned pack_bf16(float lo, float hi) {
    __bf16 a = (__bf16)lo, b = (__bf16)hi;
    return (unsigned)__builtin_bit_cast(unsigned short, a) |
           ((unsigned)__builtin_bit_cast(unsigned short, b) << 16);
}
__device__ __forceinline__ unsigned encode2(unsigned v) {
    unsigned e = v ^ 0x80008000u;
    e |= ((e & 0x0000ffffu) == 0u) ? 0x00008000u : 0u;
    e |= ((e & 0xffff0000u) == 0u) ? 0x80000000u : 0u;
    return e;
}
__device__ __forceinline__ bool present(u32x4 v) {
    bool ok = true;
#pragma unroll
    for (int j = 0; j < 4; ++j) ok = ok && (v[j] & 0xffffu) != 0u && (v[j] >> 16) != 0u;
    return ok;
}

__global__ __launch_bounds__(256, 1) void gemm_rs_kernel(Params p) {
    __shared__ __attribute__((aligned(16))) char sW[KSH * TN * 2];

    TS(0);
    const int b = blockIdx.x;
    const int tid = threadIdx.x;
    const int lane = tid & 63;
    const int wave = __builtin_amdgcn_readfirstlane(tid >> 6);
    const int wr = wave >> 1, wc = wave & 1;
    const int rank = p.rank;
    const int tn = (b & 7) * 4 + ((b >> 3) & 3);
    const int tm = b >> 5;
    const int tile = tm * (NFULL / TN) + tn;

    char* myws = p.ws[rank];
    unsigned* cnt = (unsigned*)(myws + CNT_OFF + (size_t)b * 128);
    unsigned* tmo = (unsigned*)(myws + TMO_OFF);
    const unsigned call = __hip_atomic_load(cnt, __ATOMIC_RELAXED, __HIP_MEMORY_SCOPE_AGENT) + 1u;
    const __amdgpu_buffer_rsrc_t myrs = __builtin_amdgcn_make_buffer_rsrc(myws, 0, (int)STAGE_BYTES, 0x00020000);
    const unsigned pkt = (unsigned)(tile * TILE_BYTES + wave * 1024 + lane * 16);

    const int fr = lane & 15, fg = lane >> 4;
    const int rot = b % 7;
    const uint16_t* xbase = p.x + (size_t)(tm * TM + wr * 16 + fr) * KSH + fg * 8;
    bf16x8 xf[8][8];
#define LOAD_X(i) do { \
        const int d_ = ((i) < 7) ? ((rank + 1 + (rot + (i)) % 7) & 7) : rank; \
        const uint16_t* xp_ = xbase + (size_t)d_ * MSH * KSH; \
        _Pragma("unroll") for (int ks = 0; ks < 8; ++ks) xf[i][ks] = *(const bf16x8*)(xp_ + ks * 32); \
    } while (0)
    {
        u32x4 v[8];
#pragma unroll
        for (int i = 0; i < 8; ++i) {
            const int c = tid + 256 * i, row = c >> 3, ch = c & 7;
            v[i] = *(const u32x4*)(p.w + (size_t)row * NFULL + tn * TN + ch * 8);
        }
        LOAD_X(0);
        __builtin_amdgcn_sched_barrier(0);
#pragma unroll
        for (int i = 0; i < 8; ++i) {
            const int c = tid + 256 * i, row = c >> 3, ch = c & 7;
            *(u32x4*)(sW + row * 128 + ((ch ^ (((row >> 1) & 1) << 2)) << 4)) = v[i];
        }
    }
    __syncthreads();
    TS(1);
    TSH(0);
    if (tid == 0) __hip_atomic_store(cnt, call, __ATOMIC_RELAXED, __HIP_MEMORY_SCOPE_AGENT);

    bf16x8 wf[2][8];
    {
        const int q = (lane & 15) >> 2, pp = lane & 3;
        const unsigned a0 = (unsigned)(size_t)sW + (8 * fg + q) * 128 + (((4 * wc + pp) ^ (((q >> 1) & 1) << 2)) << 4);
        bf16x4 rd[8][4];
#pragma unroll
        for (int ks = 0; ks < 8; ++ks)
            asm volatile("ds_read_b64_tr_b16 %0, %4 offset:%5\n\t"
                         "ds_read_b64_tr_b16 %1, %4 offset:%6\n\t"
                         "ds_read_b64_tr_b16 %2, %4 offset:%7\n\t"
                         "ds_read_b64_tr_b16 %3, %4 offset:%8"
                         : "=v"(rd[ks][0]), "=v"(rd[ks][1]), "=v"(rd[ks][2]), "=v"(rd[ks][3])
                         : "v"(a0), "i"(ks * 4096), "i"(ks * 4096 + 512), "i"(ks * 4096 + 8), "i"(ks * 4096 + 520)
                         : "memory");
#pragma unroll
        for (int ks = 0; ks < 8; ks += 2)
            asm volatile("s_waitcnt lgkmcnt(0)"
                         : "+v"(rd[ks][0]), "+v"(rd[ks][1]), "+v"(rd[ks][2]), "+v"(rd[ks][3]),
                           "+v"(rd[ks + 1][0]), "+v"(rd[ks + 1][1]), "+v"(rd[ks + 1][2]), "+v"(rd[ks + 1][3])
                         :: "memory");
#pragma unroll
        for (int ks = 0; ks < 8; ++ks) {
            wf[0][ks] = __builtin_shufflevector(rd[ks][0], rd[ks][1], 0, 1, 2, 3, 4, 5, 6, 7);
            wf[1][ks] = __builtin_shufflevector(rd[ks][2], rd[ks][3], 0, 1, 2, 3, 4, 5, 6, 7);
        }
    }

    const unsigned slot_off = (unsigned)(((call & 1u) * NR + rank) * SLOT_BYTES) + pkt;

    f32x4 acc0, acc1;
#pragma unroll
    for (int i = 0; i < 8; ++i) {
        __builtin_amdgcn_sched_barrier(0);
        if (i < 2) LOAD_X(i + 1);
        if (i == 2) { LOAD_X(3); LOAD_X(4); }
        if (i == 3) { LOAD_X(5); LOAD_X(6); LOAD_X(7); }
        __builtin_amdgcn_sched_barrier(0);
        acc0 = (f32x4){0.f, 0.f, 0.f, 0.f};
        acc1 = (f32x4){0.f, 0.f, 0.f, 0.f};
#pragma unroll
        for (int ks = 0; ks < 8; ++ks) {
            acc0 = __builtin_amdgcn_mfma_f32_16x16x32_bf16(wf[0][ks], xf[i][ks], acc0, 0, 0, 0);
            acc1 = __builtin_amdgcn_mfma_f32_16x16x32_bf16(wf[1][ks], xf[i][ks], acc1, 0, 0, 0);
        }
        if (i < 7) {
            const int d = (rank + 1 + (rot + i) % 7) & 7;
            u32x4 pk;
            pk[0] = encode2(pack_bf16(acc0[0], acc0[1]));
            pk[1] = encode2(pack_bf16(acc0[2], acc0[3]));
            pk[2] = encode2(pack_bf16(acc1[0], acc1[1]));
            pk[3] = encode2(pack_bf16(acc1[2], acc1[3]));
            const __amdgpu_buffer_rsrc_t rs = __builtin_amdgcn_make_buffer_rsrc(p.ws[d], 0, (int)STAGE_BYTES, 0x00020000);
            __builtin_amdgcn_raw_buffer_store_b128(pk, rs, slot_off, 0, AUX_SYS);
            if (i == 0) TSP(5);
        }
    }
    TS(2);
    TSP(6);

    u32x4 pv[7];
    bool failed = false;
    {
        const unsigned roff = (unsigned)((call & 1u) * NR * SLOT_BYTES) + pkt;
        unsigned pend = 0x7fu;
        const unsigned long long t0 = __builtin_amdgcn_s_memrealtime();
        for (unsigned spin = 0;; ++spin) {
#pragma unroll
            for (int i = 0; i < 7; ++i)
                if (pend & (1u << i))
                    pv[i] = __builtin_amdgcn_raw_buffer_load_b128(myrs, roff + (unsigned)(((rank + 1 + i) & 7) * SLOT_BYTES), 0, AUX_SYS);
#pragma unroll
            for (int i = 0; i < 7; ++i)
                if ((pend & (1u << i)) && __all(present(pv[i]))) pend &= ~(1u << i);
            if (pend == 0u) break;
            if ((spin & 63u) == 63u) {
                if (__hip_atomic_load(tmo, __ATOMIC_RELAXED, __HIP_MEMORY_SCOPE_AGENT) == call ||
                    __builtin_amdgcn_s_memrealtime() - t0 > DEADLINE_TICKS) {
                    __hip_atomic_store(tmo, call, __ATOMIC_RELAXED, __HIP_MEMORY_SCOPE_AGENT);
                    failed = true;
                    break;
                }
            }
            __builtin_amdgcn_s_sleep(1);
        }
    }
    TS(5);
    TSP(7);

    float r[8] = {acc0[0], acc0[1], acc0[2], acc0[3], acc1[0], acc1[1], acc1[2], acc1[3]};
    if (!failed) {
#pragma unroll
        for (int i = 0; i < 7; ++i) {
#pragma unroll
            for (int j = 0; j < 4; ++j) {
                const unsigned v = pv[i][j] ^ 0x80008000u;
                r[2 * j] += __builtin_bit_cast(float, v << 16);
                r[2 * j + 1] += __builtin_bit_cast(float, v & 0xffff0000u);
            }
        }
    } else {
#pragma unroll
        for (int j = 0; j < 8; ++j) r[j] = __builtin_nanf("");
    }
    u32x4 o;
    o[0] = pack_bf16(r[0], r[1]);
    o[1] = pack_bf16(r[2], r[3]);
    o[2] = pack_bf16(r[4], r[5]);
    o[3] = pack_bf16(r[6], r[7]);
    *(u32x4*)(p.out + (size_t)(tm * TM + wr * 16 + fr) * NFULL + tn * TN + wc * 32 + fg * 8) = o;
    {
        const unsigned roff = (unsigned)((call & 1u) * NR * SLOT_BYTES) + pkt;
        const u32x4 z = {0u, 0u, 0u, 0u};
#pragma unroll
        for (int i = 0; i < 7; ++i)
            __builtin_amdgcn_raw_buffer_store_b128(z, myrs, roff + (unsigned)(((rank + 1 + i) & 7) * SLOT_BYTES), 0, AUX_SYS);
    }
    TS(7);
    TSH(1);
    TSK(tstart);
}

}

extern "C" void kernel_launch(int rank, int nranks, ncclComm_t comm,
                              hipStream_t stream,
                              void* const* d_in, const int* in_sizes, int n_in,
                              void* d_out, int out_size,
                              const int* in_dtypes, int out_dtype,
                              void* const* d_out_all,
                              void* const* d_ws_all, size_t ws_bytes) {
    (void)comm; (void)in_dtypes; (void)out_dtype; (void)d_out_all;
    if (nranks != NR || (unsigned)rank >= (unsigned)NR || n_in < 2 || ws_bytes < WS_NEEDED ||
        in_sizes[0] != MFULL * KSH || in_sizes[1] != KSH * NFULL || out_size != MSH * NFULL) {
        fprintf(stderr, "gemm_rs: unexpected problem shape\n");
        return;
    }
    Params p;
    p.x = (const uint16_t*)d_in[0];
    p.w = (const uint16_t*)d_in[1];
    p.out = (uint16_t*)d_out;
    for (int r = 0; r < NR; ++r) p.ws[r] = (char*)d_ws_all[r];
    p.rank = rank;
    gemm_rs_kernel<<<dim3(NWG), dim3(256), 0, stream>>>(p);
    HIP_CHECK(hipGetLastError());
}
```

```cpp
#include <hip/hip_runtime.h>
#include <rccl/rccl.h>
#include <cstdio>
#include <cstdint>

#define HIP_CHECK(expr) do { hipError_t _e = (expr); if (_e != hipSuccess) { \
    fprintf(stderr, "HIP error %s at %s:%d\n", hipGetErrorString(_e), __FILE__, __LINE__); \
    return; } } while (0)

typedef __attribute__((ext_vector_type(8))) short bf16x8;
typedef __attribute__((ext_vector_type(4))) short bf16x4;
typedef __attribute__((ext_vector_type(4))) float f32x4;
typedef __attribute__((ext_vector_type(4))) unsigned u32x4;

namespace {

constexpr int NR = 8;
constexpr int MFULL = 2048;
constexpr int NFULL = 2048;
constexpr int KSH = 256;
constexpr int MSH = MFULL / NR;
constexpr int TM = 32, TN = 64;
constexpr int NWG = (MSH / TM) * (NFULL / TN);
constexpr int TILE_BYTES = TM * TN * 2;
constexpr size_t SLOT_BYTES = (size_t)MSH * NFULL * 2;
constexpr size_t STAGE_BYTES = 2 * NR * SLOT_BYTES;
constexpr size_t CNT_OFF = STAGE_BYTES;
constexpr size_t CNT_BYTES = (size_t)NWG * 128;
constexpr size_t TMO_OFF = CNT_OFF + CNT_BYTES;
constexpr size_t WS_NEEDED = TMO_OFF + 128;
constexpr int AUX_SYS = 17;
constexpr unsigned long long DEADLINE_TICKS = 100000000ull;

struct Params {
    const uint16_t* x;
    const uint16_t* w;
    uint16_t* out;
    char* ws[NR];
    int rank;
};

__device__ __forceinline__ unsigned pack_bf16(float lo, float hi) {
    __bf16 a = (__bf16)lo, b = (__bf16)hi;
    return (unsigned)__builtin_bit_cast(unsigned short, a) |
           ((unsigned)__builtin_bit_cast(unsigned short, b) << 16);
}
__device__ __forceinline__ unsigned encode2(unsigned v) {
    unsigned e = v ^ 0x80008000u;
    e |= ((e & 0x0000ffffu) == 0u) ? 0x00008000u : 0u;
    e |= ((e & 0xffff0000u) == 0u) ? 0x80000000u : 0u;
    return e;
}
__device__ __forceinline__ bool present(u32x4 v) {
    bool ok = true;
#pragma unroll
    for (int j = 0; j < 4; ++j) ok = ok && (v[j] & 0xffffu) != 0u && (v[j] >> 16) != 0u;
    return ok;
}

__global__ __launch_bounds__(256, 1) void gemm_rs_kernel(Params p) {
    __shared__ __attribute__((aligned(16))) char sW[KSH * TN * 2];

    const int b = blockIdx.x;
    const int tid = threadIdx.x;
    const int lane = tid & 63;
    const int wave = __builtin_amdgcn_readfirstlane(tid >> 6);
    const int wr = wave >> 1, wc = wave & 1;
    const int rank = p.rank;
    const int tn = (b & 7) * 4 + ((b >> 3) & 3);
    const int tm = b >> 5;
    const int tile = tm * (NFULL / TN) + tn;

    char* myws = p.ws[rank];
    unsigned* cnt = (unsigned*)(myws + CNT_OFF + (size_t)b * 128);
    unsigned* tmo = (unsigned*)(myws + TMO_OFF);
    const unsigned call = __hip_atomic_load(cnt, __ATOMIC_RELAXED, __HIP_MEMORY_SCOPE_AGENT) + 1u;
    const __amdgpu_buffer_rsrc_t myrs = __builtin_amdgcn_make_buffer_rsrc(myws, 0, (int)STAGE_BYTES, 0x00020000);
    const unsigned pkt = (unsigned)(tile * TILE_BYTES + wave * 1024 + lane * 16);

    const int fr = lane & 15, fg = lane >> 4;
    const int rot = b % 7;
    const uint16_t* xbase = p.x + (size_t)(tm * TM + wr * 16 + fr) * KSH + fg * 8;
    bf16x8 xf[8][8];
#define LOAD_X(i) do { \
        const int d_ = ((i) < 7) ? ((rank + 1 + (rot + (i)) % 7) & 7) : rank; \
        const uint16_t* xp_ = xbase + (size_t)d_ * MSH * KSH; \
        _Pragma("unroll") for (int ks = 0; ks < 8; ++ks) xf[i][ks] = *(const bf16x8*)(xp_ + ks * 32); \
    } while (0)
    {
        u32x4 v[8];
#pragma unroll
        for (int i = 0; i < 8; ++i) {
            const int c = tid + 256 * i, row = c >> 3, ch = c & 7;
            v[i] = *(const u32x4*)(p.w + (size_t)row * NFULL + tn * TN + ch * 8);
        }
        LOAD_X(0);
        __builtin_amdgcn_sched_barrier(0);
#pragma unroll
        for (int i = 0; i < 8; ++i) {
            const int c = tid + 256 * i, row = c >> 3, ch = c & 7;
            *(u32x4*)(sW + row * 128 + ((ch ^ (((row >> 1) & 1) << 2)) << 4)) = v[i];
        }
    }
    __syncthreads();
    if (tid == 0) __hip_atomic_store(cnt, call, __ATOMIC_RELAXED, __HIP_MEMORY_SCOPE_AGENT);

    bf16x8 wf[2][8];
    {
        const int q = (lane & 15) >> 2, pp = lane & 3;
        const unsigned a0 = (unsigned)(size_t)sW + (8 * fg + q) * 128 + (((4 * wc + pp) ^ (((q >> 1) & 1) << 2)) << 4);
        bf16x4 rd[8][4];
#pragma unroll
        for (int ks = 0; ks < 8; ++ks)
            asm volatile("ds_read_b64_tr_b16 %0, %4 offset:%5\n\t"
                         "ds_read_b64_tr_b16 %1, %4 offset:%6\n\t"
                         "ds_read_b64_tr_b16 %2, %4 offset:%7\n\t"
                         "ds_read_b64_tr_b16 %3, %4 offset:%8"
                         : "=v"(rd[ks][0]), "=v"(rd[ks][1]), "=v"(rd[ks][2]), "=v"(rd[ks][3])
                         : "v"(a0), "i"(ks * 4096), "i"(ks * 4096 + 512), "i"(ks * 4096 + 8), "i"(ks * 4096 + 520)
                         : "memory");
#pragma unroll
        for (int ks = 0; ks < 8; ks += 2)
            asm volatile("s_waitcnt lgkmcnt(0)"
                         : "+v"(rd[ks][0]), "+v"(rd[ks][1]), "+v"(rd[ks][2]), "+v"(rd[ks][3]),
                           "+v"(rd[ks + 1][0]), "+v"(rd[ks + 1][1]), "+v"(rd[ks + 1][2]), "+v"(rd[ks + 1][3])
                         :: "memory");
#pragma unroll
        for (int ks = 0; ks < 8; ++ks) {
            wf[0][ks] = __builtin_shufflevector(rd[ks][0], rd[ks][1], 0, 1, 2, 3, 4, 5, 6, 7);
            wf[1][ks] = __builtin_shufflevector(rd[ks][2], rd[ks][3], 0, 1, 2, 3, 4, 5, 6, 7);
        }
    }

    const unsigned slot_off = (unsigned)(((call & 1u) * NR + rank) * SLOT_BYTES) + pkt;

    f32x4 acc0, acc1;
#pragma unroll
    for (int i = 0; i < 8; ++i) {
        __builtin_amdgcn_sched_barrier(0);
        if (i < 2) LOAD_X(i + 1);
        if (i == 2) { LOAD_X(3); LOAD_X(4); }
        if (i == 3) { LOAD_X(5); LOAD_X(6); LOAD_X(7); }
        __builtin_amdgcn_sched_barrier(0);
        acc0 = (f32x4){0.f, 0.f, 0.f, 0.f};
        acc1 = (f32x4){0.f, 0.f, 0.f, 0.f};
#pragma unroll
        for (int ks = 0; ks < 8; ++ks) {
            acc0 = __builtin_amdgcn_mfma_f32_16x16x32_bf16(wf[0][ks], xf[i][ks], acc0, 0, 0, 0);
            acc1 = __builtin_amdgcn_mfma_f32_16x16x32_bf16(wf[1][ks], xf[i][ks], acc1, 0, 0, 0);
        }
        if (i < 7) {
            const int d = (rank + 1 + (rot + i) % 7) & 7;
            u32x4 pk;
            pk[0] = encode2(pack_bf16(acc0[0], acc0[1]));
            pk[1] = encode2(pack_bf16(acc0[2], acc0[3]));
            pk[2] = encode2(pack_bf16(acc1[0], acc1[1]));
            pk[3] = encode2(pack_bf16(acc1[2], acc1[3]));
            const __amdgpu_buffer_rsrc_t rs = __builtin_amdgcn_make_buffer_rsrc(p.ws[d], 0, (int)STAGE_BYTES, 0x00020000);
            __builtin_amdgcn_raw_buffer_store_b128(pk, rs, slot_off, 0, AUX_SYS);
        }
    }

    u32x4 pv[7];
    bool failed = false;
    {
        const unsigned roff = (unsigned)((call & 1u) * NR * SLOT_BYTES) + pkt;
        unsigned pend = 0x7fu;
        const unsigned long long t0 = __builtin_amdgcn_s_memrealtime();
        for (unsigned spin = 0;; ++spin) {
            asm volatile("" ::: "memory");
#pragma unroll
            for (int i = 0; i < 7; ++i)
                if (pend & (1u << i))
                    pv[i] = __builtin_amdgcn_raw_buffer_load_b128(myrs, roff + (unsigned)(((rank + 1 + i) & 7) * SLOT_BYTES), 0, AUX_SYS);
#pragma unroll
            for (int i = 0; i < 7; ++i)
                if ((pend & (1u << i)) && __all(present(pv[i]))) pend &= ~(1u << i);
            if (pend == 0u) break;
            if ((spin & 63u) == 63u) {
                if (__hip_atomic_load(tmo, __ATOMIC_RELAXED, __HIP_MEMORY_SCOPE_AGENT) == call ||
                    __builtin_amdgcn_s_memrealtime() - t0 > DEADLINE_TICKS) {
                    __hip_atomic_store(tmo, call, __ATOMIC_RELAXED, __HIP_MEMORY_SCOPE_AGENT);
                    failed = true;
                    break;
                }
            }
            __builtin_amdgcn_s_sleep(1);
        }
    }

    float r[8] = {acc0[0], acc0[1], acc0[2], acc0[3], acc1[0], acc1[1], acc1[2], acc1[3]};
    if (!failed) {
#pragma unroll
        for (int i = 0; i < 7; ++i) {
#pragma unroll
            for (int j = 0; j < 4; ++j) {
                const unsigned v = pv[i][j] ^ 0x80008000u;
                r[2 * j] += __builtin_bit_cast(float, v << 16);
                r[2 * j + 1] += __builtin_bit_cast(float, v & 0xffff0000u);
            }
        }
    } else {
#pragma unroll
        for (int j = 0; j < 8; ++j) r[j] = __builtin_nanf("");
    }
    u32x4 o;
    o[0] = pack_bf16(r[0], r[1]);
    o[1] = pack_bf16(r[2], r[3]);
    o[2] = pack_bf16(r[4], r[5]);
    o[3] = pack_bf16(r[6], r[7]);
    *(u32x4*)(p.out + (size_t)(tm * TM + wr * 16 + fr) * NFULL + tn * TN + wc * 32 + fg * 8) = o;
    {
        const unsigned roff = (unsigned)((call & 1u) * NR * SLOT_BYTES) + pkt;
        const u32x4 z = {0u, 0u, 0u, 0u};
#pragma unroll
        for (int i = 0; i < 7; ++i)
            __builtin_amdgcn_raw_buffer_store_b128(z, myrs, roff + (unsigned)(((rank + 1 + i) & 7) * SLOT_BYTES), 0, AUX_SYS);
    }
}

}

extern "C" void kernel_launch(int rank, int nranks, ncclComm_t comm,
                              hipStream_t stream,
                              void* const* d_in, const int* in_sizes, int n_in,
                              void* d_out, int out_size,
                              const int* in_dtypes, int out_dtype,
                              void* const* d_out_all,
                              void* const* d_ws_all, size_t ws_bytes) {
    (void)comm; (void)in_dtypes; (void)out_dtype; (void)d_out_all;
    if (nranks != NR || (unsigned)rank >= (unsigned)NR || n_in < 2 || ws_bytes < WS_NEEDED ||
        in_sizes[0] != MFULL * KSH || in_sizes[1] != KSH * NFULL || out_size != MSH * NFULL) {
        fprintf(stderr, "gemm_rs: unexpected problem shape\n");
        return;
    }
    Params p;
    p.x = (const uint16_t*)d_in[0];
    p.w = (const uint16_t*)d_in[1];
    p.out = (uint16_t*)d_out;
    for (int r = 0; r < NR; ++r) p.ws[r] = (char*)d_ws_all[r];
    p.rank = rank;
    gemm_rs_kernel<<<dim3(NWG), dim3(256), 0, stream>>>(p);
    HIP_CHECK(hipGetLastError());
}
```
